```python
import numpy as np

kernel_name = "gpu_gemm_ar_m4096_k4096_n4096_w8a8_dqsilu"


ABSMAX_RTOL = 0.001

COMPUTE_DTYPE = "int8"
EPILOGUE = "dequant_silu"
IN_QUANT = "int8"
OUT_QUANT = None
IN_DTYPES = {"x": "int8", "w_mat": "int8",
             "scale_x": "float32", "scale_w": "float32"}
WORLD_SIZE = 8

def setup_inputs():

    x = np.random.randint(-127, 128, size=(4096, 4096)).astype(np.int8)
    w_mat = np.random.randint(-127, 128, size=(4096, 4096)).astype(np.int8)
    scale_x = (0.02 * (1.0 + np.random.rand(1))).astype(np.float32)
    scale_w = (0.02 * (1.0 + np.random.rand(1))).astype(np.float32)
    return {"x": x, "w_mat": w_mat, "scale_x": scale_x, "scale_w": scale_w}

def reference(x, w_mat, scale_x, scale_w):
    acc = x.astype(np.float64) @ w_mat.astype(np.float64)
    y = (acc.astype(np.float32) * (scale_x * scale_w)).astype(np.float32)
    return (y / (1.0 + np.exp(-np.clip(y, -60.0, 60.0)))).astype(np.float32)
```

```cpp
#include <hip/hip_runtime.h>
#include <rccl/rccl.h>
#include <cstdio>

#define HIP_CHECK(expr) do { hipError_t _e = (expr); if (_e != hipSuccess) { \
    fprintf(stderr, "HIP error %s at %s:%d\n", hipGetErrorString(_e), __FILE__, __LINE__); \
    return; } } while (0)

namespace gar {
#define LAS __attribute__((address_space(3)))
#define GAS __attribute__((address_space(1)))
typedef GAS unsigned gu32;
typedef int i32x4 __attribute__((ext_vector_type(4)));
typedef unsigned u32x4 __attribute__((ext_vector_type(4)));
typedef float f32x4 __attribute__((ext_vector_type(4)));
typedef float f32x2 __attribute__((ext_vector_type(2)));
typedef unsigned long long u64;

constexpr int NR = 8;
constexpr int MM = 4096, NN = 4096, KS = 512;
constexpr int NPUSH = 16, NWG = 256, NGEMM = NWG - NPUSH;
constexpr int NUNIT = 512;
constexpr int NQ = NUNIT / NPUSH;
constexpr int NDIRECT = 4;
constexpr int HTB = 16384;
constexpr int NKT = 32;
constexpr size_t KSTEP = (size_t)32 * HTB;
constexpr size_t OPB = (size_t)16 << 20;
constexpr size_t SLOTB = 2 * OPB;
constexpr size_t FLAG_OFF = 2 * SLOTB;
constexpr size_t CNT_OFF = FLAG_OFF + 4096 * 128;
constexpr size_t TMO_OFF = CNT_OFF + (size_t)NWG * 128;
constexpr size_t WS_NEED = TMO_OFF + 4096;
constexpr int LDS_FLAGS = 9 * HTB;
constexpr int LDS_CNT = LDS_FLAGS + 2048;
constexpr int LDS_RDY = LDS_CNT + 128;
constexpr int LDS_BYTES = LDS_CNT + 256;
constexpr u64 DEADLINE = 40000000ull;
constexpr int PACE = 226;
constexpr int T0 = 190;

struct Schedule { unsigned char v[NR][128]; };
constexpr Schedule kSchedule = {{
    {1, 5, 72, 76, 65, 69, 10, 14, 73, 77, 9, 13, 0, 4, 8, 12, 64, 68, 2, 6, 74, 78, 26, 30, 66, 70, 67, 71, 3, 7, 50, 54, 57, 61, 17, 21, 75, 79, 114, 118, 11, 15, 82, 86, 18, 22, 98, 102, 121, 125, 122, 126, 89, 93, 120, 124, 97, 101, 56, 60, 40, 44, 81, 85,
     34, 38, 90, 94, 105, 109, 25, 29, 24, 28, 88, 92, 16, 20, 104, 108, 80, 84, 106, 110, 112, 116, 32, 36, 113, 117, 33, 37, 58, 62, 42, 46, 123, 127, 96, 100, 41, 45, 51, 55, 49, 53, 59, 63, 48, 52, 19, 23, 43, 47, 83, 87, 91, 95, 35, 39, 27, 31, 115, 119, 99, 103, 107, 111},
    {3, 7, 11, 15, 67, 71, 8, 12, 72, 76, 64, 68, 2, 6, 50, 54, 74, 78, 10, 14, 75, 79, 0, 4, 27, 31, 66, 70, 9, 13, 107, 111, 106, 110, 18, 22, 114, 118, 58, 62, 1, 5, 73, 77, 65, 69, 90, 94, 99, 103, 105, 109, 26, 30, 33, 37, 83, 87, 98, 102, 17, 21, 82, 86,
     97, 101, 49, 53, 81, 85, 42, 46, 43, 47, 122, 126, 91, 95, 25, 29, 57, 61, 115, 119, 48, 52, 34, 38, 123, 127, 121, 125, 51, 55, 59, 63, 113, 117, 56, 60, 19, 23, 41, 45, 89, 93, 35, 39, 96, 100, 104, 108, 16, 20, 112, 116, 120, 124, 40, 44, 32, 36, 24, 28, 88, 92, 80, 84},
    {72, 76, 64, 68, 8, 12, 73, 77, 0, 4, 2, 6, 50, 54, 66, 70, 75, 79, 1, 5, 11, 15, 74, 78, 3, 7, 24, 28, 67, 71, 56, 60, 10, 14, 112, 116, 9, 13, 104, 108, 65, 69, 48, 52, 41, 45, 34, 38, 32, 36, 123, 127, 120, 124, 43, 47, 57, 61, 40, 44, 113, 117, 16, 20,
     82, 86, 25, 29, 96, 100, 18, 22, 51, 55, 80, 84, 98, 102, 115, 119, 88, 92, 114, 118, 90, 94, 42, 46, 91, 95, 19, 23, 35, 39, 26, 30, 33, 37, 58, 62, 59, 63, 27, 31, 83, 87, 122, 126, 97, 101, 107, 111, 106, 110, 89, 93, 99, 103, 17, 21, 121, 125, 49, 53, 81, 85, 105, 109},
    {75, 79, 73, 77, 11, 15, 65, 69, 1, 5, 66, 70, 9, 13, 67, 71, 50, 54, 0, 4, 8, 12, 64, 68, 81, 85, 74, 78, 72, 76, 122, 126, 41, 45, 3, 7, 10, 14, 2, 6, 34, 38, 56, 60, 82, 86, 57, 61, 106, 110, 98, 102, 88, 92, 90, 94, 97, 101, 26, 30, 104, 108, 32, 36,
     89, 93, 16, 20, 120, 124, 40, 44, 112, 116, 24, 28, 42, 46, 114, 118, 113, 117, 25, 29, 18, 22, 58, 62, 80, 84, 33, 37, 105, 109, 121, 125, 49, 53, 48, 52, 115, 119, 96, 100, 83, 87, 43, 47, 17, 21, 35, 39, 27, 31, 107, 111, 51, 55, 99, 103, 123, 127, 59, 63, 19, 23, 91, 95},
    {10, 14, 2, 6, 74, 78, 66, 70, 0, 4, 97, 101, 3, 7, 65, 69, 75, 79, 33, 37, 9, 13, 73, 77, 64, 68, 90, 94, 8, 12, 11, 15, 98, 102, 50, 54, 1, 5, 105, 109, 72, 76, 67, 71, 25, 29, 113, 117, 49, 53, 18, 22, 43, 47, 106, 110, 122, 126, 82, 86, 121, 125, 27, 31,
     89, 93, 17, 21, 57, 61, 107, 111, 42, 46, 123, 127, 34, 38, 58, 62, 114, 118, 81, 85, 19, 23, 51, 55, 26, 30, 83, 87, 91, 95, 99, 103, 35, 39, 115, 119, 112, 116, 59, 63, 41, 45, 88, 92, 24, 28, 120, 124, 56, 60, 40, 44, 80, 84, 104, 108, 48, 52, 16, 20, 32, 36, 96, 100},
    {66, 70, 3, 7, 75, 79, 2, 6, 74, 78, 10, 14, 97, 101, 11, 15, 41, 45, 33, 37, 67, 71, 65, 69, 73, 77, 121, 125, 105, 109, 1, 5, 9, 13, 17, 21, 49, 53, 8, 12, 89, 93, 0, 4, 114, 118, 72, 76, 64, 68, 59, 63, 34, 38, 91, 95, 42, 46, 106, 110, 25, 29, 81, 85,
     120, 124, 57, 61, 58, 62, 16, 20, 43, 47, 18, 22, 113, 117, 122, 126, 50, 54, 26, 30, 98, 102, 123, 127, 51, 55, 35, 39, 99, 103, 83, 87, 107, 111, 27, 31, 82, 86, 32, 36, 19, 23, 88, 92, 104, 108, 90, 94, 115, 119, 48, 52, 112, 116, 24, 28, 80, 84, 40, 44, 56, 60, 96, 100},
    {56, 60, 9, 13, 74, 78, 65, 69, 64, 68, 1, 5, 0, 4, 73, 77, 114, 118, 66, 70, 82, 86, 8, 12, 2, 6, 10, 14, 72, 76, 34, 38, 11, 15, 32, 36, 98, 102, 75, 79, 106, 110, 42, 46, 67, 71, 3, 7, 96, 100, 50, 54, 104, 108, 16, 20, 122, 126, 120, 124, 90, 94, 58, 62,
     26, 30, 35, 39, 40, 44, 18, 22, 91, 95, 88, 92, 51, 55, 25, 29, 99, 103, 27, 31, 112, 116, 48, 52, 107, 111, 59, 63, 33, 37, 43, 47, 24, 28, 19, 23, 115, 119, 123, 127, 113, 117, 105, 109, 83, 87, 80, 84, 121, 125, 97, 101, 41, 45, 17, 21, 57, 61, 89, 93, 49, 53, 81, 85},
    {0, 4, 73, 77, 1, 5, 65, 69, 72, 76, 8, 12, 75, 79, 9, 13, 3, 7, 64, 68, 10, 14, 67, 71, 112, 116, 11, 15, 27, 31, 74, 78, 2, 6, 66, 70, 35, 39, 123, 127, 99, 103, 51, 55, 122, 126, 26, 30, 24, 28, 40, 44, 16, 20, 115, 119, 104, 108, 98, 102, 90, 94, 34, 38,
     50, 54, 42, 46, 58, 62, 80, 84, 82, 86, 107, 111, 83, 87, 91, 95, 59, 63, 114, 118, 96, 100, 43, 47, 106, 110, 48, 52, 19, 23, 18, 22, 25, 29, 49, 53, 121, 125, 97, 101, 17, 21, 120, 124, 32, 36, 33, 37, 56, 60, 88, 92, 81, 85, 57, 61, 113, 117, 105, 109, 41, 45, 89, 93},
}};
struct Tables {
    unsigned char piece[NR][4][NQ];
    unsigned char when[NR][128];
};
constexpr Tables make_tables(const Schedule& G) {
    Tables t = {};
    for (int r = 0; r < NR; ++r) for (int i = 0; i < 128; ++i) { t.piece[r][i & 3][i >> 2] = G.v[r][i]; t.when[r][G.v[r][i] & 127] = (unsigned char)i; }
    return t;
}
__constant__ const Tables kTables = make_tables(kSchedule);

struct Params {
    char* peer[NR];
    const char* x; const char* w; const float* sx; const float* sw; float* out;
    int rank; int pad;
    unsigned first[4];
};
__host__ __device__ __forceinline__ int job_of(int g, int which) {
    const int x = g & 7, k = g >> 3, base = (x >> 1) * 16 + 2 * (x & 1);
    if (k < 4) return which == 0 ? ((base + (k >> 1)) | ((1 + (k & 1)) << 8)) : base + 4 * (1 + (k >> 1)) + (k & 1);
    if (which) return -1;
    const int n = k - 4;
    if (n < 2) return base + 12 + n;
    return base + 64 * (1 + ((n - 2) >> 3)) + 4 * (((n - 2) & 7) >> 1) + (n & 1);
}
__host__ __device__ __forceinline__ int piece_code(int piece, int quarter) {
    return ((piece >> 6) << 8) | (((piece >> 2) & 15) << 4) | ((quarter >> 1) << 3) | ((piece & 3) << 1) | (quarter & 1);
}

__device__ __forceinline__ void st_sys(char* p, u32x4 v) {
    asm volatile("global_store_dwordx4 %0, %1, off sc0 sc1\n\ts_nop 1" :: "v"(p), "v"(v) : "memory");
}
__device__ __forceinline__ void glds_sys(const void* gsrc, unsigned lds_dst) {
    asm volatile("s_mov_b32 m0, %1\n\ts_nop 0\n\tglobal_load_lds_dwordx4 %0, off sc0 sc1" :: "v"(gsrc), "s"(lds_dst) : "memory");
}
__device__ __forceinline__ void flag_store(char* p, unsigned val, u64 m) {
    u64 sv;
    asm volatile("s_mov_b64 %0, exec\n\ts_mov_b64 exec, %3\n\tglobal_store_dword %1, %2, off sc0 sc1\n\ts_mov_b64 exec, %0"
                 : "=&s"(sv) : "v"(p), "v"(val), "s"(m) : "memory");
}
template <class T> __device__ __forceinline__ T* uni(T* q) {
    const u64 a = (u64)q;
    const unsigned lo = (unsigned)__builtin_amdgcn_readfirstlane((int)(unsigned)a), hi = (unsigned)__builtin_amdgcn_readfirstlane((int)(unsigned)(a >> 32));
    return (T*)(((u64)hi << 32) | lo);
}
#define WAIT_V(n) asm volatile("s_waitcnt vmcnt(" #n ")" ::: "memory")
__device__ __forceinline__ void lds_publish(LAS volatile int* w, int v) { asm volatile("" ::: "memory"); *w = v; }
__device__ __forceinline__ int lds_await(LAS volatile int* w, int need) {
    int v;
    while ((v = __builtin_amdgcn_readfirstlane(*w)) <= need) __builtin_amdgcn_s_sleep(1);
    asm volatile("" ::: "memory");
    return v;
}
#define WAIT_L(n) asm volatile("s_waitcnt lgkmcnt(" #n ")" ::: "memory")
#define BAR __builtin_amdgcn_s_barrier()
#define SCHED __builtin_amdgcn_sched_barrier(0)

__device__ __forceinline__ unsigned frag_off(int rank, int code, int v) {
    const int o = code >> 8, blk = (code >> 4) & 15, h = (code >> 3) & 1, s = (code >> 1) & 3, u = code & 1;
    const unsigned piece = (unsigned)o * (unsigned)OPB + (unsigned)(((rank * 4 + s) * 32 + blk * 2) * HTB);
    return piece + (o == 0 ? (unsigned)(h * HTB + u * 8192 + v * 1024)
                           : (unsigned)((v >> 2) * HTB + (((h * 2 + u) * 2 + ((v >> 1) & 1)) * 2 + (v & 1)) * 1024));
}
__device__ __forceinline__ int flag_off(int rank, int code) {
    const int o = code >> 8, blk = (code >> 4) & 15, h = (code >> 3) & 1, s = (code >> 1) & 3, u = code & 1;
    return (((o * 16 + blk) * 32 + rank * 4 + s) * 4 + h * 2 + u) * 128;
}
struct Raw { u32x4 r[4]; };
__device__ __forceinline__ Raw load_raw(const Params& p, int code, int v, int lane) {
    const int o = code >> 8, blk = (code >> 4) & 15, h = (code >> 3) & 1, s = (code >> 1) & 3, u = code & 1;
    const char* a = (o == 0) ? p.x + (size_t)(blk * 256 + h * 128 + (4 * u + (v >> 1)) * 16 + (lane & 15)) * KS + s * 128 + (v & 1) * 64 + (lane >> 4) * 16
                             : p.w + (size_t)(s * 128 + (v & 1) * 64 + (lane >> 2)) * NN + blk * 256 + (h * 2 + u) * 64 + (lane & 3) * 16;
    Raw w;
    w.r[0] = *(const u32x4*)a;
#pragma unroll
    for (int i = 1; i < 4; ++i) w.r[i] = *(const u32x4*)((o == 0) ? p.x : a + (size_t)i * 16 * NN);
    return w;
}
constexpr int TSCR = 4 * 1088;
__device__ __forceinline__ u32x4 finish_fragment(const Raw& w, int code, int v, int lane, LAS char* tl) {
    if ((code >> 8) == 0) return w.r[0];
#pragma unroll
    for (int i = 0; i < 4; ++i) *(LAS u32x4*)(tl + i * 1088 + lane * 16) = w.r[i];
    WAIT_L(0);
    const LAS unsigned char* rp = (const LAS unsigned char*)(tl + (lane >> 4) * 1088 + (lane & 15) * 4 + (v >> 1));
    unsigned r[4];
#pragma unroll
    for (int d = 0; d < 4; ++d)
        r[d] = (unsigned)rp[(d * 4) * 64] | ((unsigned)rp[(d * 4 + 1) * 64] << 8) | ((unsigned)rp[(d * 4 + 2) * 64] << 16) | ((unsigned)rp[(d * 4 + 3) * 64] << 24);
    u32x4 val = (u32x4){r[0], r[1], r[2], r[3]};
    asm volatile("s_waitcnt lgkmcnt(0)" : "+v"(val) :: "memory");
    return val;
}
__device__ __forceinline__ bool give_up(char* myws, unsigned call, u64 t0, unsigned& spins) {
    gu32* tmo = (gu32*)(myws + TMO_OFF);
    bool dead = __builtin_amdgcn_s_memrealtime() - t0 > DEADLINE;
    if (!dead && (++spins & 31u) == 0u)
        dead = (unsigned)__builtin_amdgcn_readfirstlane(__hip_atomic_load(tmo, __ATOMIC_RELAXED, __HIP_MEMORY_SCOPE_AGENT)) == call;
    if (dead) { __hip_atomic_store(tmo, call, __ATOMIC_RELAXED, __HIP_MEMORY_SCOPE_AGENT); WAIT_V(0); }
    return dead;
}

__device__ __forceinline__ unsigned load_call(gu32* cntp) {
    return (unsigned)__builtin_amdgcn_readfirstlane((int)__hip_atomic_load(cntp, __ATOMIC_RELAXED, __HIP_MEMORY_SCOPE_AGENT)) + 1u;
}
__device__ __forceinline__ unsigned pusher(const Params& p, char* myws, gu32* cntp, LAS char* lds, int lane, int v, u64 tstart) {
    const int b = blockIdx.x, rank = p.rank;
    LAS unsigned* cnt = (LAS unsigned*)(lds + LDS_CNT);
    if (threadIdx.x < NQ) cnt[threadIdx.x] = 0;
    if (threadIdx.x == NQ) *(LAS int*)(lds + LDS_RDY) = 0;
    LAS char* tl = lds + v * (4096 + TSCR);
    const unsigned tl_addr = (unsigned)(size_t)tl;
    const unsigned f4 = p.first[b >> 2];
    u64 fc = 0;
#pragma unroll
    for (int q = 0; q < NDIRECT; ++q) fc |= (u64)piece_code((f4 >> (8 * q)) & 255, b & 3) << (16 * q);
    Raw raw[NDIRECT];
    raw[0] = load_raw(p, (int)fc & 0xffff, v, lane);
    unsigned callv = __hip_atomic_load(cntp, __ATOMIC_RELAXED, __HIP_MEMORY_SCOPE_AGENT);
    asm volatile("" ::: "memory");
#pragma unroll
    for (int q = 1; q < NDIRECT; ++q) raw[q] = load_raw(p, (int)(fc >> (16 * q)) & 0xffff, v, lane);
    int mc = piece_code((int)kTables.piece[rank][b >> 2][lane & (NQ - 1)], b & 3);
    char* wp[NR];
#pragma unroll
    for (int i = 0; i < NR; ++i) wp[i] = uni(p.peer[i]);
    char* fb = p.peer[0];
#pragma unroll
    for (int i = 1; i < NR - 1; ++i) fb = (lane == i) ? p.peer[i] : fb;
    fb += FLAG_OFF;
    asm volatile("" : "+v"(raw[0].r[0]), "+v"(raw[0].r[1]), "+v"(raw[0].r[2]), "+v"(raw[0].r[3]), "+v"(callv));
    const unsigned call = (unsigned)__builtin_amdgcn_readfirstlane((int)callv) + 1u;
    const unsigned slot = (call & 1u) * (unsigned)SLOTB;
#pragma unroll
    for (int i = 0; i < NR; ++i) wp[i] += slot;
    u32x4 cur = finish_fragment(raw[0], (int)fc & 0xffff, v, lane, tl + 4096);
    u64 tnext = tstart + (u64)(T0 + b * PACE / NPUSH);
    while (__builtin_amdgcn_s_memrealtime() < tnext) __builtin_amdgcn_s_sleep(1);
    {
        const unsigned off = frag_off(rank, (int)fc & 0xffff, v) + lane * 16;
#pragma unroll
        for (int i = 0; i < NR - 1; ++i) st_sys(wp[i] + off, cur);
    }
#pragma unroll
    for (int q = 1; q < NDIRECT; ++q) *(LAS u32x4*)(tl + q * 1024 + lane * 16) = finish_fragment(raw[q], (int)(fc >> (16 * q)) & 0xffff, v, lane, tl + 4096);
    WAIT_L(0);
    BAR;
    for (int q = 0; q < NQ; ++q) {
        if (q > 0) {
            WAIT_V(1);
            if (q == 1 && v == 0) {
                gu32* lf = (gu32*)(myws + FLAG_OFF + flag_off(rank, mc));
                const u64 t0 = __builtin_amdgcn_s_memrealtime(); unsigned spins = 0;
                while (!__all((int)(__hip_atomic_load(lf, __ATOMIC_RELAXED, __HIP_MEMORY_SCOPE_SYSTEM) - call) >= 0)) {
                    if (give_up(myws, call, t0, spins)) break;
                    __builtin_amdgcn_s_sleep(4);
                }
                lds_publish((LAS volatile int*)(lds + LDS_RDY), 1);
            }
            unsigned old = 0;
            if (lane == 0) old = __hip_atomic_fetch_add(cnt + (q - 1), 1u, __ATOMIC_RELAXED, __HIP_MEMORY_SCOPE_WORKGROUP);
            const bool last = (unsigned)__builtin_amdgcn_readfirstlane((int)old) == 7u;
            flag_store(fb + flag_off(rank, __builtin_amdgcn_readlane(mc, q - 1)), call, last ? 0x7full : 0ull);
            tnext += PACE;
            while (__builtin_amdgcn_s_memrealtime() < tnext) __builtin_amdgcn_s_sleep(1);
            const unsigned off = frag_off(rank, __builtin_amdgcn_readlane(mc, q), v) + lane * 16;
#pragma unroll
            for (int i = 0; i < NR - 1; ++i) st_sys(wp[i] + off, cur);
        }
        if (q == NDIRECT - 2) (void)lds_await((LAS volatile int*)(lds + LDS_RDY), 0);
        const int ql = (q + 2 >= NDIRECT && q + 2 < NQ) ? q + 2 : q;
        glds_sys(wp[7] + frag_off(rank, __builtin_amdgcn_readlane(mc, ql), v) + lane * 16, tl_addr + ((ql == q) ? 4096u : (unsigned)((q + 2) & 3) * 1024u));
        if (q > 0) WAIT_V(9);
        cur = *(const LAS u32x4*)(tl + ((q + 1) & 3) * 1024 + lane * 16);
        asm volatile("s_waitcnt lgkmcnt(0)" : "+v"(cur) :: "memory");
    }
    WAIT_V(0);
    {
        unsigned old = 0;
        if (lane == 0) old = __hip_atomic_fetch_add(cnt + (NQ - 1), 1u, __ATOMIC_RELAXED, __HIP_MEMORY_SCOPE_WORKGROUP);
        const bool last = (unsigned)__builtin_amdgcn_readfirstlane((int)old) == 7u;
        flag_store(fb + flag_off(rank, __builtin_amdgcn_readlane(mc, NQ - 1)), call, last ? 0x7full : 0ull);
    }
    return call;
}

struct Job { int pm, pn, hsel, ordv; const char* cA; const char* cB; float scale; };
__device__ __forceinline__ Job make_job(const Params& p, int job, const char* slot, int lane) {
    Job j;
    const int tile = job & 255;
    j.pm = __builtin_amdgcn_readfirstlane(tile >> 4); j.pn = __builtin_amdgcn_readfirstlane(tile & 15); j.hsel = __builtin_amdgcn_readfirstlane((job >> 9) & 1);
    const int kt = lane & 31, r = kt >> 2;
    const int ra = kTables.when[r][j.pm * 4 + (kt & 3)], rb = kTables.when[r][64 + j.pn * 4 + (kt & 3)];
    const int key = ((r == p.rank ? 0 : 1 + (ra > rb ? ra : rb)) << 10) | (((kt * 7) & 31) << 5) | kt;
    int pos = 0;
#pragma unroll
    for (int k = 0; k < NKT; ++k) pos += (__builtin_amdgcn_readlane(key, k) < key) ? 1 : 0;
    j.ordv = __builtin_amdgcn_ds_permute((pos + (lane & 32)) * 4, kt);
    j.cA = slot + (size_t)(2 * j.pm + j.hsel) * HTB;
    j.cB = slot + OPB + (size_t)(2 * j.pn) * HTB;
    j.scale = p.sx[0] * p.sw[0];
    return j;
}

__device__ __forceinline__ unsigned stager(const Params& p, char* myws, gu32* cntp, LAS char* lds, int lane, int v, Job& J0, Job& J1) {
    const int g = blockIdx.x - NPUSH, rank = p.rank;
    int code[3];
    Raw raw[3];
#pragma unroll
    for (int k = 0; k < 3; ++k) {
        const int i = g + k * NGEMM;
        code[k] = (i < NUNIT) ? (((1 - ((i >> 4) & 1)) << 8) | ((i >> 5) << 4) | (((i >> 1) & 1) << 3) | (((i >> 2) & 3) << 1) | (i & 1)) : -1;
        if (code[k] >= 0) raw[k] = load_raw(p, code[k], v, lane);
    }
    const unsigned call = load_call(cntp);
    char* base = myws + (call & 1u) * (unsigned)SLOTB + lane * 16;
#pragma unroll
    for (int k = 0; k < 3; ++k)
        if (code[k] >= 0) st_sys(base + frag_off(rank, code[k], v), finish_fragment(raw[k], code[k], v, lane, lds + v * TSCR));
    J0 = make_job(p, job_of(g, 0), myws, lane);
    J1 = make_job(p, job_of(g, 1) >= 0 ? job_of(g, 1) : job_of(g, 0), myws, lane);
    if (threadIdx.x < 2) ((LAS int*)(lds + LDS_RDY))[threadIdx.x] = 0;
    WAIT_L(0); WAIT_V(0); BAR;
    if (threadIdx.x < 3) {
        const int c = threadIdx.x == 0 ? code[0] : threadIdx.x == 1 ? code[1] : code[2];
        if (c >= 0) __hip_atomic_store((gu32*)(myws + FLAG_OFF + flag_off(rank, c)), call, __ATOMIC_RELAXED, __HIP_MEMORY_SCOPE_SYSTEM);
    }
    return call;
}

__device__ __forceinline__ const char* window_addr(const char* fbase, int ordv, int wb, int lane, int pm, int pn) {
    int idx = wb + (lane >> 3); idx = idx > NKT - 1 ? NKT - 1 : idx;
    const int kt = __builtin_amdgcn_ds_bpermute(idx * 4, ordv);
    const int o = (lane >> 2) & 1;
    return fbase + (size_t)((((o * 16 + (o ? pn : pm)) * 32 + kt) * 4 + (lane & 3)) * 128);
}
__device__ __forceinline__ int window_count(unsigned f, unsigned call, int wb) {
    const u64 miss = ~__builtin_amdgcn_ballot_w64((int)(f - call) >= 0);
    const int n = wb + (miss == 0ull ? 8 : (__builtin_ctzll(miss) >> 3));
    return n > NKT ? NKT : n;
}
__device__ __forceinline__ int ready_poll(const char* a, unsigned call, int wb) {
    unsigned f;
    asm volatile("global_load_dword %0, %1, off sc0 sc1\n\ts_waitcnt vmcnt(0)" : "=&v"(f) : "v"(a) : "memory");
    return window_count(f, call, wb);
}
__device__ __forceinline__ void glds4_sys(const void* gsrc, unsigned lds_dst) {
    asm volatile("s_mov_b32 m0, %1\n\ts_nop 0\n\tglobal_load_lds_dword %0, off sc0 sc1" :: "v"(gsrc), "s"(lds_dst) : "memory");
}
__device__ __forceinline__ void st_out(float* p, f32x4 v) {
    asm volatile("global_store_dwordx4 %0, %1, off sc1\n\ts_nop 1" :: "v"(p), "v"(v) : "memory");
}
__device__ __forceinline__ f32x2 silu2(int a0, int a1, f32x2 s2, f32x2 c2) {
    const f32x2 a = (f32x2){(float)a0, (float)a1};
    const f32x2 y = a * s2;
    f32x2 t = a * c2;
    t.x = __builtin_amdgcn_fmed3f(t.x, -86.56170245f, 86.56170245f);
    t.y = __builtin_amdgcn_fmed3f(t.y, -86.56170245f, 86.56170245f);
    f32x2 d = (f32x2){__builtin_amdgcn_exp2f(t.x), __builtin_amdgcn_exp2f(t.y)} + 1.0f;
    return y * (f32x2){__builtin_amdgcn_rcpf(d.x), __builtin_amdgcn_rcpf(d.y)};
}

__device__ __forceinline__ void gemm_tiles(const Params& p, char* myws, unsigned call, LAS char* lds, int lane, int wid, Job J, const Job& J1, bool two) {
    const int tid = threadIdx.x;
    const int wr = wid >> 2, wc = wid & 3;
    const unsigned slot = (call & 1u) * (unsigned)SLOTB;
    const unsigned voff = tid * 16, voff2 = tid * 16 + 8192;
    const unsigned ldsb = (unsigned)(size_t)lds + wid * 1024;
    const int aoff = wr * 8192 + lane * 16, boff = wc * 4096 + lane * 16;
    LAS char* fl = lds + LDS_FLAGS + wid * 256;
    const unsigned fl_addr = (unsigned)(size_t)fl;
    const char* fbase = myws + FLAG_OFF;
#define STAGE(bufoff, gbase) asm volatile("s_mov_b32 m0, %3\n\ts_nop 0\n\tglobal_load_lds_dwordx4 %0, %2 sc0 sc1\n\ts_add_u32 m0, m0, 0x2000\n\ts_nop 0\n\tglobal_load_lds_dwordx4 %1, %2 sc0 sc1" \
        :: "v"(voff), "v"(voff2), "s"(gbase), "s"(ldsb + (unsigned)(bufoff)) : "memory", "scc")
#define LDA(dst, off) do { _Pragma("unroll") for (int m = 0; m < 4; ++m) _Pragma("unroll") for (int k = 0; k < 2; ++k) dst[m][k] = *(const LAS i32x4*)(lds + (off) + aoff + m * 2048 + k * 1024); } while (0)
#define LDB(dst, off) do { _Pragma("unroll") for (int n = 0; n < 2; ++n) _Pragma("unroll") for (int k = 0; k < 2; ++k) dst[n][k] = *(const LAS i32x4*)(lds + (off) + boff + n * 2048 + k * 1024); } while (0)
#define MMA(ai, bj, At, Bt) do { __builtin_amdgcn_s_setprio(1); _Pragma("unroll") for (int m = 0; m < 4; ++m) _Pragma("unroll") for (int n = 0; n < 2; ++n) _Pragma("unroll") for (int k = 0; k < 2; ++k) \
        acc[ai][bj][m][n] = __builtin_amdgcn_mfma_i32_16x16x64_i8(At[m][k], Bt[n][k], acc[ai][bj][m][n], 0, 0, 0); __builtin_amdgcn_s_setprio(0); } while (0)
#define NEED(J, need) do { if (__builtin_expect(nready <= (need), 0)) { if (wid == 0) { const u64 _t0 = __builtin_amdgcn_s_memrealtime(); unsigned _sp = 0; \
        for (;;) { nready = ready_poll(window_addr(fbase, J.ordv, nready, lane, J.pm, J.pn), call, nready); if (nready > (need)) break; \
                   if (give_up(myws, call, _t0, _sp)) { nready = NKT; break; } __builtin_amdgcn_s_sleep(2); } \
        lds_publish(rdy, nready); } else { nready = lds_await(rdy, (need)); } } } while (0)
#define PEEK_ISSUE(J) const int wb = nready; const bool refresh = wid == 0 && wb < NKT; \
        if (refresh) glds4_sys(window_addr(fbase, J.ordv, wb, lane, J.pm, J.pn), fl_addr)
#define PEEK_READ() if (refresh) { unsigned f = *(const LAS unsigned*)(fl + lane * 4); asm volatile("s_waitcnt lgkmcnt(0)" : "+v"(f) :: "memory"); \
        const int r = window_count(f, call, wb); if (r > nready) { nready = r; lds_publish(rdy, r); } }
#define KOFF(J, i) ((size_t)__builtin_amdgcn_readlane(J.ordv, (i) & 31) * KSTEP)
#define ZERO_ACC() do { _Pragma("unroll") for (int a = 0; a < 2; ++a) _Pragma("unroll") for (int bb = 0; bb < 2; ++bb) _Pragma("unroll") for (int m = 0; m < 4; ++m) \
        _Pragma("unroll") for (int n = 0; n < 2; ++n) acc[a][bb][m][n] = (i32x4){0, 0, 0, 0}; } while (0)
#define EPILOGUE(J, NAI) do { \
        const unsigned tmo = (unsigned)__builtin_amdgcn_readfirstlane((int)*(const LAS unsigned*)fl); \
        const float s = (tmo == call) ? __builtin_nanf("") : J.scale;        \
        const float c1 = -s * 1.4426950408889634f; \
        const f32x2 s2 = (f32x2){s, s}, c2 = (f32x2){c1, c1}; \
        int tz = threadIdx.x; asm volatile("" : "+v"(tz)); \
        const int l2 = tz & 63, fr = l2 & 15, fq = l2 >> 4; \
        float* obase = p.out + (size_t)(J.pm * 256 + J.hsel * 128 + wr * 64 + 4 * fq) * NN + J.pn * 256 + wc * 64 + 4 * fr; \
        _Pragma("unroll") for (int ai = 0; ai < NAI; ++ai) _Pragma("unroll") for (int m = 0; m < 4; ++m) _Pragma("unroll") for (int j = 0; j < 4; ++j) { \
            const f32x2 lo = silu2(acc[ai][0][m][0][j], acc[ai][0][m][1][j], s2, c2), hi = silu2(acc[ai][1][m][0][j], acc[ai][1][m][1][j], s2, c2); \
            st_out(obase + (size_t)(ai * 128 + m * 16 + j) * NN, (f32x4){lo.x, lo.y, hi.x, hi.y}); } \
    } while (0)
#define FB(b, h) (((b) * 2 + (h)) * HTB)
#define FA1(b) ((4 + (b)) * HTB)
#define FULL_STAGE0(J) do { const size_t k0 = KOFF(J, 0), k1 = KOFF(J, 1); \
        STAGE(FB(0, 0), J.cB + k0); STAGE(FB(0, 1), J.cB + k0 + HTB); STAGE(6 * HTB, J.cA + k0); STAGE(FA1(0), J.cA + k0 + HTB); \
        STAGE(FB(1, 0), J.cB + k1); STAGE(FB(1, 1), J.cB + k1 + HTB); STAGE(7 * HTB, J.cA + k1); } while (0)

    i32x4 acc[2][2][4][2];
    i32x4 At[4][2], B0[2][2], B1[2][2];
    int nready = 0;
    LAS volatile int* rdy = (LAS volatile int*)(lds + LDS_RDY);
    J.cA = uni(J.cA + slot); J.cB = uni(J.cB + slot);
    if (two) {
        ZERO_ACC();
        NEED(J, 1);
        {
            const size_t k0 = KOFF(J, 0), k1 = KOFF(J, 1);
            STAGE(0, J.cB + k0); STAGE(2 * HTB, J.cA + k0); STAGE(HTB, J.cB + k0 + HTB);
            STAGE(3 * HTB, J.cB + k1); STAGE(5 * HTB, J.cA + k1); STAGE(4 * HTB, J.cB + k1 + HTB);
        }
        if (wr == 1) BAR;
        WAIT_V(8); BAR; BAR;
        int cur = 0, nx2 = 6 * HTB;
        const char *a2, *b2;
        { const size_t n2 = KOFF(J, 2); a2 = J.cA + n2; b2 = J.cB + n2; }
#define HALF_STEP(MORE, WA, WB) do { \
            LDB(B0, cur); SCHED; LDA(At, cur + 2 * HTB); \
            if (MORE) { NEED(J, t + 2); STAGE(nx2, b2); STAGE(nx2 + 2 * HTB, a2); } \
            WAIT_V(WA); \
            if (MORE && (t & 1) == 0) { PEEK_READ(); }          \
            WAIT_L(8); BAR; WAIT_L(0); MMA(0, 0, At, B0); BAR; SCHED; \
            LDB(B1, cur + HTB); \
            if (MORE) { STAGE(nx2 + HTB, b2 + HTB); const size_t n3 = KOFF(J, t + 3); a2 = J.cA + n3; b2 = J.cB + n3; } \
            WAIT_V(WB); BAR; WAIT_L(0); MMA(0, 1, At, B1); BAR; SCHED; \
            nx2 = cur; cur = (cur == 6 * HTB) ? 0 : cur + 3 * HTB; \
        } while (0)
        int t = 0;
        bool refresh = false; int wb = 0;
        for (; t < NKT - 2; ++t) {
            HALF_STEP(true, 10, 8);
            if ((t & 1) == 0) {
                wb = nready; refresh = wid == 0 && wb < NKT && t + 2 < NKT - 2;
                if (refresh) glds4_sys(window_addr(fbase, J.ordv, wb, lane, J.pm, J.pn), fl_addr);
            }
        }
        HALF_STEP(false, 6, 2);
        ++t;
        glds4_sys(myws + TMO_OFF, fl_addr);
        HALF_STEP(false, 0, 0);
        if (wr == 0) BAR;
        const Job H = J;
        J = J1; J.cA = uni(J.cA + slot); J.cB = uni(J.cB + slot);
        nready = 0; ++rdy;
        NEED(J, 1);
        FULL_STAGE0(J);
        EPILOGUE(H, 1);
        if (wr == 1) BAR;
        WAIT_V(24); BAR; BAR;
    } else {
        NEED(J, 1);
        FULL_STAGE0(J);
        if (wr == 1) BAR;
        WAIT_V(8); BAR; BAR;
    }
    ZERO_ACC();
    J.cA = uni(J.cA); J.cB = uni(J.cB); J.pm = __builtin_amdgcn_readfirstlane(J.pm); J.pn = __builtin_amdgcn_readfirstlane(J.pn);
#define KSTEP(b, S1, S2, W1, W2) do { \
          \
        LDB(B0, FB(b, 0)); LDB(B1, FB(b, 1)); SCHED; LDA(At, ar0); \
        if (S1) STAGE(FA1(1 - (b)), a1 + HTB); \
        else glds4_sys(myws + TMO_OFF, fl_addr);                \
        WAIT_V(W1); \
        WAIT_L(8); BAR; WAIT_L(0); MMA(0, 0, At, B0); MMA(0, 1, At, B1); BAR; SCHED; \
          \
        LDA(At, FA1(b)); \
        if (S2) { NEED(J, t + 2); \
            STAGE(ar2, a2); STAGE(FB(b, 0), b2); STAGE(FB(b, 1), b2 + HTB); \
            a1 = a2; { const size_t n3 = KOFF(J, t + 3); a2 = J.cA + n3; b2 = J.cB + n3; } } \
        { const int x = ar0; ar0 = ar1; ar1 = ar2; ar2 = x; } \
        WAIT_V(W2); \
        WAIT_L(0); BAR; MMA(1, 0, At, B0); MMA(1, 1, At, B1); BAR; SCHED; \
        ++t; \
    } while (0)
#define KSTEP1(b) KSTEP(b, true, true, 8, 8)
    {
        int t = 0, ar0 = 6 * HTB, ar1 = 7 * HTB, ar2 = 8 * HTB;
        const char* a1 = J.cA + KOFF(J, 1);
        const char *a2, *b2;
        { const size_t n2 = KOFF(J, 2); a2 = J.cA + n2; b2 = J.cB + n2; }
        while (t < NKT - 2) {
            PEEK_ISSUE(J);
            KSTEP1(0);
            KSTEP1(1);
            PEEK_READ();
        }
        KSTEP(0, true, false, 8, 2);
        KSTEP(1, false, false, 0, 0);
    }
    if (wr == 0) BAR;
    EPILOGUE(J, 2);
}

__global__ void __launch_bounds__(512, 2) gemm_ar_kernel(Params p) {
    extern __shared__ __attribute__((aligned(16))) char shm[];
    LAS char* lds = (LAS char*)shm;
    const int lane = threadIdx.x & 63, wid = __builtin_amdgcn_readfirstlane(threadIdx.x >> 6);
    char* myws = uni(p.peer[NR - 1]);
    gu32* cntp = (gu32*)(myws + CNT_OFF + (size_t)blockIdx.x * 128);
    const u64 tstart = __builtin_amdgcn_s_memrealtime();
    unsigned call;
    if (blockIdx.x < NPUSH) {
        call = pusher(p, myws, cntp, lds, lane, wid, tstart);
    } else {
        Job J0, J1;
        call = stager(p, myws, cntp, lds, lane, wid, J0, J1);
        const bool two = job_of(blockIdx.x - NPUSH, 1) >= 0;
        gemm_tiles(p, myws, call, lds, lane, wid, J0, J1, two);
    }
    WAIT_V(0); BAR;
    if (threadIdx.x == 0) __hip_atomic_store(cntp, call, __ATOMIC_RELAXED, __HIP_MEMORY_SCOPE_AGENT);
}

}

extern "C" void kernel_launch(int rank, int nranks, ncclComm_t comm, hipStream_t stream,
                              void* const* d_in, const int* in_sizes, int n_in,
                              void* d_out, int out_size,
                              const int* in_dtypes, int out_dtype,
                              void* const* d_out_all,
                              void* const* d_ws_all, size_t ws_bytes) {
    using namespace gar;
    (void)comm; (void)in_sizes; (void)in_dtypes; (void)out_dtype; (void)d_out_all;
    if (nranks != NR || (unsigned)rank >= (unsigned)NR || n_in != 4 || out_size != MM * NN || ws_bytes < WS_NEED) {
        fprintf(stderr, "kernel_launch: unexpected problem shape\n");
        return;
    }
    HIP_CHECK(hipFuncSetAttribute((const void*)gemm_ar_kernel, hipFuncAttributeMaxDynamicSharedMemorySize, LDS_BYTES));
    Params p;
    for (int i = 0; i < NR; ++i) p.peer[i] = (char*)d_ws_all[(rank + 1 + i) & 7];
    p.x = (const char*)d_in[0]; p.w = (const char*)d_in[1]; p.sx = (const float*)d_in[2]; p.sw = (const float*)d_in[3];
    p.out = (float*)d_out; p.rank = rank; p.pad = 0;
    const Schedule& sch = kSchedule;
    for (int k = 0; k < 4; ++k) { p.first[k] = 0; for (int q = 0; q < 4; ++q) p.first[k] |= (unsigned)sch.v[rank][q * 4 + k] << (8 * q); }
    hipLaunchKernelGGL(gemm_ar_kernel, dim3(NWG), dim3(512), LDS_BYTES, stream, p);
    HIP_CHECK(hipGetLastError());
}
```
